```python
import numpy as np

kernel_name = "gpu_gemm_ar_m4096_k4096_n32768_bf16_gelu"


ABSMAX_RTOL = 0.008

COMPUTE_DTYPE = "bfloat16"
EPILOGUE = "gelu"
IN_QUANT = None
OUT_QUANT = None
IN_DTYPES = {"x": "bfloat16", "w_mat": "bfloat16"}
WORLD_SIZE = 8

def _to_bf16(x):
    u = np.ascontiguousarray(x, dtype=np.float32).view(np.uint32)
    return ((u + 0x7FFF + ((u >> 16) & 1)) & 0xFFFF0000).view(np.float32)

def setup_inputs():

    x = _to_bf16(np.random.randn(4096, 4096).astype(np.float32))
    w_mat = _to_bf16((np.random.randn(4096, 32768) / np.sqrt(4096)).astype(np.float32))
    return {"x": x, "w_mat": w_mat}

def _epilogue(y):
    c = 0.7978845608028654
    return (0.5 * y * (1.0 + np.tanh(c * (y + 0.044715 * y ** 3)))).astype(np.float32)

def reference(x, w_mat):
    y = (x @ w_mat).astype(np.float32)
    return _epilogue(y)
```

```cpp
#include <hip/hip_runtime.h>
#include <rccl/rccl.h>
#include <cstdio>
#include <cstdint>

#define HIP_CHECK(expr) do { hipError_t _e = (expr); if (_e != hipSuccess) { \
    fprintf(stderr, "HIP error %s at %s:%d\n", hipGetErrorString(_e), __FILE__, __LINE__); \
    return; } } while (0)

namespace {

constexpr int NCOPYWG = 128;
constexpr int WPL = 18;
constexpr int HEAD_RAW = 20;
constexpr int TAIL_RAW = 16;
#define WIN 3
constexpr unsigned RUNAHEAD = 2;


typedef unsigned u32;
typedef unsigned short u16;
typedef unsigned long long u64;
typedef unsigned u32x4 __attribute__((ext_vector_type(4)));
typedef unsigned u32x2 __attribute__((ext_vector_type(2)));
typedef float f32x4 __attribute__((ext_vector_type(4)));
typedef short bf16x8 __attribute__((ext_vector_type(8)));
typedef short s16x4 __attribute__((ext_vector_type(4)));
typedef unsigned short u16x2 __attribute__((ext_vector_type(2)));
#define LAS __attribute__((address_space(3)))
#define GAS __attribute__((address_space(1)))
typedef __amdgpu_buffer_rsrc_t rsrc_t;

constexpr int NR = 8;
constexpr int M = 4096, KT = 4096, NTOT = 32768, KS = 512, NS = 4096;
constexpr int BM = 256, BN = 128, BK = 64, NKT = KT / BK;
constexpr int NWG = 256, NTHR = 512;
constexpr int NGEMMWG = NWG - NCOPYWG;
static_assert(NGEMMWG == 128, "16 GEMM workgroups per XCD group");
static_assert(7 * WPL <= NCOPYWG, "copy geometry");
static_assert(HEAD_RAW % 4 == 0 && TAIL_RAW % 4 == 0 && HEAD_RAW <= 64, "raw units come in whole jobs / tiles");
constexpr u32 USLOT = 16384;
constexpr int NXU = 256, NWU = 256, NIU = NXU + NWU, NOU = 2048, NU = NIU + NOU;
constexpr int NJOBS = 16 * 36;

constexpr size_t OFF_CALL = 0, OFF_TMO = 128, OFF_F3 = 4096;
constexpr size_t OFF_TK = 16384;
constexpr size_t OFF_F1 = 1u << 20;
constexpr size_t F1_STRIDE = 128;
constexpr size_t OFF_A = 8u << 20, OFF_B = 40u << 20;
constexpr size_t OFF_ENC = 72u << 20;
constexpr size_t ENC_X = 0, ENC_W = 4u << 20, ENC_O = 36u << 20, ENC_PAR = 68u << 20;
constexpr size_t WS_NEED = OFF_ENC + 2 * ENC_PAR;
static_assert(OFF_F1 + 2 * 8 * NU * F1_STRIDE <= OFF_A, "flag area");
constexpr u64 DEADLINE = 150000000ull;

constexpr int LDS_STAGE = (BM + BN) * BK * 2;
constexpr int LDS_RING = 3 * LDS_STAGE;
constexpr int L_ENC = LDS_RING;
constexpr int L_MISC = L_ENC + 8192;
constexpr int L_XCH = L_MISC + 512;
constexpr int L_DUMMY = L_XCH + 2048;
constexpr int LDS_BYTES = L_DUMMY + 256;
constexpr int EP_STRIDE = 272;
constexpr int WT_STRIDE = 144;
constexpr int L_SCR_EP = BM * EP_STRIDE;
constexpr int L_SCR_W = 512 * WT_STRIDE;
static_assert(L_SCR_EP + 8 * 8192 <= LDS_RING && L_SCR_W + 8 * 8192 <= LDS_RING, "lds");
constexpr int L_DEC_OUT = 0, L_DEC_IN = 8192, L_DEC_ID = 16384, L_SLOT = 24576, NSLOT = 8;
static_assert(L_SLOT + NSLOT * 16384 <= L_MISC, "lds");
constexpr int L_OK = L_MISC, L_CALL = L_MISC + 4, L_FULL = L_MISC + 64, L_INFO = L_MISC + 128, L_SIDX = L_MISC + 192, L_FREE = L_MISC + 256;
constexpr int L_QFULL = L_MISC + 320, L_QINFO = L_MISC + 352, L_QS = L_MISC + 384, L_QUSED = L_MISC + 416;

constexpr int AUX_SYS = 17;
constexpr int AUX_DEV = 16;

constexpr int N_IN = 296;
constexpr unsigned short SYM_IN[N_IN] = {
    0x371c, 0x373c, 0x378c, 0x379c, 0x37ac, 0x37bc, 0x37cc, 0x37dc, 0x37ec, 0x37fc, 0x380c, 0x381c, 0x382c, 0x383c, 0x384c, 0x385c,
    0x386c, 0x387c, 0x388c, 0x389c, 0x38ac, 0x38bc, 0x38cc, 0x38dc, 0x38ec, 0x38fc, 0x390c, 0x391c, 0x392c, 0x393c, 0x394c, 0x395c,
    0x396c, 0x397c, 0x398c, 0x399c, 0x39ac, 0x39bc, 0x39cc, 0x39dc, 0x39ec, 0x39fc, 0x3a0c, 0x3a1c, 0x3a2c, 0x3a3c, 0x3a4c, 0x3a5c,
    0x3a6c, 0x3a7c, 0x3a8c, 0x3a9c, 0x3aac, 0x3abc, 0x3acc, 0x3adc, 0x3aec, 0x3afc, 0x3b0c, 0x3b1c, 0x3b2c, 0x3b3c, 0x3b4c, 0x3b5c,
    0x3b6c, 0x3b7c, 0x3b8c, 0x3b9c, 0x3bac, 0x3bbc, 0x3bcc, 0x3bdc, 0x3bec, 0x3bfc, 0x3c0b, 0x3c1b, 0x3c2b, 0x3c3b, 0x3c4b, 0x3c5b,
    0x3c6b, 0x3c7b, 0x3c8a, 0x3c9a, 0x3caa, 0x3cba, 0x3cca, 0x3cda, 0x3cea, 0x3cfa, 0x3d09, 0x3d19, 0x3d29, 0x3d39, 0x3d49, 0x3d59,
    0x3d69, 0x3d79, 0x3d89, 0x3d98, 0x3da8, 0x3db8, 0x3dc8, 0x3dd8, 0x3de8, 0x3df8, 0x3e07, 0x3e17, 0x3e27, 0x3e37, 0x3e47, 0x3e57,
    0x3e67, 0x3e77, 0x3e87, 0x3e96, 0x3ea7, 0x3eb7, 0x3ec7, 0x3ed7, 0x3ee7, 0x3ef7, 0x3f06, 0x3f16, 0x3f26, 0x3f36, 0x3f46, 0x3f56,
    0x3f66, 0x3f76, 0x3f85, 0x3f95, 0x3fa6, 0x3fb6, 0x3fc6, 0x3fd6, 0x3fe7, 0x3ff7, 0x4007, 0x4017, 0x4028, 0x4039, 0x404a, 0x405b,
    0x406c, 0x407c, 0x408c, 0x409c, 0x40ac, 0xb78c, 0xb79c, 0xb7ac, 0xb7bc, 0xb7cc, 0xb7dc, 0xb7ec, 0xb7fc, 0xb80c, 0xb81c, 0xb82c,
    0xb83c, 0xb84c, 0xb85c, 0xb86c, 0xb87c, 0xb88c, 0xb89c, 0xb8ac, 0xb8bc, 0xb8cc, 0xb8dc, 0xb8ec, 0xb8fc, 0xb90c, 0xb91c, 0xb92c,
    0xb93c, 0xb94c, 0xb95c, 0xb96c, 0xb97c, 0xb98c, 0xb99c, 0xb9ac, 0xb9bc, 0xb9cc, 0xb9dc, 0xb9ec, 0xb9fc, 0xba0c, 0xba1c, 0xba2c,
    0xba3c, 0xba4c, 0xba5c, 0xba6c, 0xba7c, 0xba8c, 0xba9c, 0xbaac, 0xbabc, 0xbacc, 0xbadc, 0xbaec, 0xbafc, 0xbb0c, 0xbb1c, 0xbb2c,
    0xbb3c, 0xbb4c, 0xbb5c, 0xbb6c, 0xbb7c, 0xbb8c, 0xbb9c, 0xbbac, 0xbbbc, 0xbbcc, 0xbbdc, 0xbbec, 0xbbfc, 0xbc0b, 0xbc1b, 0xbc2b,
    0xbc3b, 0xbc4b, 0xbc5b, 0xbc6b, 0xbc7b, 0xbc8a, 0xbc9a, 0xbcaa, 0xbcba, 0xbcca, 0xbcda, 0xbcea, 0xbcfa, 0xbd09, 0xbd19, 0xbd29,
    0xbd39, 0xbd49, 0xbd59, 0xbd69, 0xbd79, 0xbd88, 0xbd98, 0xbda8, 0xbdb8, 0xbdc8, 0xbdd8, 0xbde8, 0xbdf8, 0xbe07, 0xbe17, 0xbe27,
    0xbe37, 0xbe47, 0xbe57, 0xbe67, 0xbe78, 0xbe86, 0xbe96, 0xbea7, 0xbeb7, 0xbec7, 0xbed7, 0xbee7, 0xbef7, 0xbf06, 0xbf16, 0xbf26,
    0xbf36, 0xbf46, 0xbf56, 0xbf66, 0xbf76, 0xbf85, 0xbf95, 0xbfa6, 0xbfb6, 0xbfc6, 0xbfd6, 0xbfe7, 0xbff7, 0xc007, 0xc017, 0xc028,
    0xc039, 0xc04a, 0xc05b, 0xc06c, 0xc07c, 0xc08c, 0xc09c, 0xc0ac,
};
constexpr int N_OUT = 280;
constexpr unsigned short SYM_OUT[N_OUT] = {
    0x370c, 0x371c, 0x372c, 0x373c, 0x374c, 0x375c, 0x376c, 0x377c, 0x378c, 0x379c, 0x37ac, 0x37bc, 0x37cc, 0x37dc, 0x37ec, 0x37fc,
    0x380c, 0x381c, 0x382c, 0x383c, 0x384c, 0x385c, 0x386c, 0x387c, 0x388c, 0x389c, 0x38ac, 0x38bc, 0x38cc, 0x38dc, 0x38ec, 0x38fc,
    0x390c, 0x391c, 0x392c, 0x393c, 0x394c, 0x395c, 0x396c, 0x397c, 0x398c, 0x399c, 0x39ac, 0x39bc, 0x39cc, 0x39dc, 0x39ec, 0x39fc,
    0x3a0c, 0x3a1c, 0x3a2c, 0x3a3c, 0x3a4c, 0x3a5c, 0x3a6c, 0x3a7c, 0x3a8c, 0x3a9c, 0x3aac, 0x3abc, 0x3acc, 0x3adc, 0x3aec, 0x3afc,
    0x3b0c, 0x3b1c, 0x3b2c, 0x3b3c, 0x3b4c, 0x3b5c, 0x3b6c, 0x3b7c, 0x3b8c, 0x3b9b, 0x3bab, 0x3bbb, 0x3bcb, 0x3bdb, 0x3beb, 0x3bfb,
    0x3c0a, 0x3c1a, 0x3c2a, 0x3c3a, 0x3c4a, 0x3c5a, 0x3c6a, 0x3c7a, 0x3c89, 0x3c99, 0x3ca9, 0x3cb9, 0x3cc9, 0x3cd9, 0x3ce9, 0x3cfa,
    0x3d09, 0x3d19, 0x3d29, 0x3d39, 0x3d49, 0x3d59, 0x3d69, 0x3d79, 0x3d88, 0x3d98, 0x3da8, 0x3db8, 0x3dc8, 0x3dd8, 0x3de8, 0x3df8,
    0x3e07, 0x3e17, 0x3e27, 0x3e37, 0x3e47, 0x3e57, 0x3e67, 0x3e77, 0x3e86, 0x3e96, 0x3ea6, 0x3eb6, 0x3ec6, 0x3ed6, 0x3ee7, 0x3ef7,
    0x3f06, 0x3f16, 0x3f26, 0x3f36, 0x3f46, 0x3f56, 0x3f66, 0x3f76, 0x3f86, 0x3f96, 0x3fa6, 0x3fb6, 0x3fc6, 0x3fd7, 0x3fe7, 0x3ff7,
    0x4007, 0x4017, 0x4028, 0x4039, 0x404a, 0x405b, 0x406c, 0x407c, 0x408c, 0x409c, 0x40ac, 0xb63c, 0xb64c, 0xb68c, 0xb69c, 0xb6ac,
    0xb6bc, 0xb6cc, 0xb6dc, 0xb6ec, 0xb6fc, 0xb70c, 0xb71c, 0xb72c, 0xb73c, 0xb74c, 0xb75c, 0xb76c, 0xb77c, 0xb78c, 0xb79c, 0xb7ac,
    0xb7bc, 0xb7cc, 0xb7dc, 0xb7ec, 0xb7fc, 0xb80c, 0xb81c, 0xb82c, 0xb83c, 0xb84c, 0xb85c, 0xb86c, 0xb87c, 0xb88c, 0xb89c, 0xb8ac,
    0xb8bc, 0xb8cc, 0xb8dc, 0xb8ec, 0xb8fc, 0xb90c, 0xb91c, 0xb92c, 0xb93c, 0xb94c, 0xb95c, 0xb96c, 0xb97c, 0xb98c, 0xb99c, 0xb9ac,
    0xb9bc, 0xb9cc, 0xb9dc, 0xb9ec, 0xb9fc, 0xba0c, 0xba1c, 0xba2c, 0xba3c, 0xba4c, 0xba5c, 0xba6c, 0xba7c, 0xba8c, 0xba9c, 0xbaac,
    0xbabc, 0xbacc, 0xbadc, 0xbaec, 0xbafc, 0xbb0c, 0xbb1c, 0xbb2c, 0xbb3c, 0xbb4c, 0xbb5c, 0xbb6c, 0xbb7c, 0xbb8b, 0xbb9b, 0xbbab,
    0xbbbb, 0xbbcb, 0xbbdb, 0xbbeb, 0xbbfb, 0xbc0a, 0xbc1a, 0xbc2a, 0xbc3a, 0xbc4a, 0xbc5a, 0xbc6a, 0xbc7a, 0xbc89, 0xbc99, 0xbca9,
    0xbcb9, 0xbcc9, 0xbcd9, 0xbce9, 0xbcf8, 0xbd07, 0xbd17, 0xbd27, 0xbd37, 0xbd47, 0xbd57, 0xbd67, 0xbd77, 0xbd86, 0xbd96, 0xbda6,
    0xbdb6, 0xbdc6, 0xbdd6, 0xbde6, 0xbdf6, 0xbe04, 0xbe14, 0xbe23,
};
struct Tables { u16 enc_in[4096], enc_out[4096], dec_in[4096], dec_out[4096]; };
constexpr void build_code(const unsigned short* sl, int n, u16* enc, u16* dec) {
    u32 code = 0;
    for (int i = 0; i < 4096; ++i) { enc[i] = 0; dec[i] = 0; }
    for (u32 len = 1; len <= 12; ++len) {
        for (int i = 0; i < n; ++i) {
            if ((sl[i] & 15u) != len) continue;
            u32 rev = 0;
            for (u32 b = 0; b < len; ++b) rev |= ((code >> b) & 1u) << (len - 1 - b);
            const u32 sym = sl[i] >> 4;
            enc[sym] = (u16)((rev << 4) | len);
            for (u32 j = rev; j < 4096u; j += 1u << len) dec[j] = (u16)((sym << 4) | len);
            ++code;
        }
        code <<= 1;
    }
}
constexpr Tables make_tables() {
    Tables t{};
    build_code(SYM_IN, N_IN, t.enc_in, t.dec_in);
    build_code(SYM_OUT, N_OUT, t.enc_out, t.dec_out);
    return t;
}
__constant__ const __attribute__((aligned(16))) Tables TAB = make_tables();

struct Params {
    const unsigned short* x;
    const unsigned short* w;
    unsigned short* out;
    void* const* ws_all;
    int rank;
    int pad;
};

struct Order { unsigned char slot[64]; unsigned short job[576]; };
constexpr Order make_order() {
    Order o{};
    int n = 0;
    auto X = [&](int p) { o.slot[n++] = (unsigned char)(2 * p); o.slot[n++] = (unsigned char)(2 * p + 1); };
    auto W = [&](int p) { o.slot[n++] = (unsigned char)(32 + p); };
    for (int i = 0; i < 4; ++i) X(i);
    for (int i = 0; i < 4; ++i) W(i);
    for (int i = 4; i < 8; ++i) W(i);
    for (int i = 4; i < 8; ++i) X(i);
    for (int i = 8; i < 16; ++i) W(i);
    for (int i = 8; i < 12; ++i) X(i);
    for (int i = 16; i < 24; ++i) W(i);
    for (int i = 12; i < 16; ++i) X(i);
    for (int i = 24; i < 32; ++i) W(i);
    int j = 0;
    for (int q = 0; q < 64; ++q) {
        if (o.slot[q] < 32) { for (int sub = 0; sub < 2; ++sub) o.job[j++] = (unsigned short)(q << 8 | 8 << 1 | sub); }
        else { for (int sub = 0; sub < 2; ++sub) for (int d = 0; d < 8; ++d) o.job[j++] = (unsigned short)(q << 8 | d << 1 | sub); }
    }
    return o;
}
constexpr Order ORD_H = make_order();
struct OrderW { unsigned slot[64]; unsigned job[576]; };
constexpr OrderW make_orderw() { OrderW w{}; for (int i = 0; i < 64; ++i) w.slot[i] = ORD_H.slot[i]; for (int i = 0; i < 576; ++i) w.job[i] = ORD_H.job[i] | (unsigned)ORD_H.slot[ORD_H.job[i] >> 8] << 16; return w; }
__constant__ const OrderW ORD = make_orderw();

struct GroupList { unsigned v[32]; };
__constant__ const GroupList GROUPS = {{0x00, 0x10, 0x01, 0x11, 0x20, 0x21, 0x30, 0x31,
                                        0x02, 0x12, 0x22, 0x32, 0x40, 0x41, 0x50, 0x51,
                                        0x42, 0x52, 0x03, 0x13, 0x23, 0x33, 0x43, 0x53,
                                        0x60, 0x61, 0x62, 0x63, 0x70, 0x71, 0x72, 0x73}};
__device__ __forceinline__ void tile_of(unsigned k, unsigned& ti, unsigned& tj) {
    const unsigned g = GROUPS.v[k >> 4];
    ti = 4u * (g & 15u) + ((k >> 2) & 3u); tj = 4u * (g >> 4) + (k & 3u);
}

__device__ __forceinline__ u32 rfl(u32 v) { return __builtin_amdgcn_readfirstlane(v); }
__device__ __forceinline__ rsrc_t mk_rsrc(const void* p, u32 bytes) {
    const u64 a = (u64)p;
    void* q = (void*)(((u64)rfl((u32)(a >> 32)) << 32) | rfl((u32)a));
    return __builtin_amdgcn_make_buffer_rsrc(q, 0, rfl(bytes), 0x00020000);
}
__device__ __forceinline__ u32 perm(u32 hi, u32 lo, u32 sel) { return __builtin_amdgcn_perm(hi, lo, sel); }
__device__ __forceinline__ u32 bfi(u32 m, u32 a, u32 b) { return (m & a) | (~m & b); }
__device__ __forceinline__ u32 pk_add16(u32 a, u32 b) { return __builtin_bit_cast(u32, (u16x2)(__builtin_bit_cast(u16x2, a) + __builtin_bit_cast(u16x2, b))); }
__device__ __forceinline__ u32 pk_sub16(u32 a, u32 b) { return __builtin_bit_cast(u32, (u16x2)(__builtin_bit_cast(u16x2, a) - __builtin_bit_cast(u16x2, b))); }
#define DPP0(x, ctrl, rmask) ((u32)__builtin_amdgcn_update_dpp(0, (int)(x), ctrl, rmask, 0xf, false))
__device__ __forceinline__ u32 wave_scan_add(u32 x) {
    x += DPP0(x, 0x111, 0xf); x += DPP0(x, 0x112, 0xf); x += DPP0(x, 0x114, 0xf); x += DPP0(x, 0x118, 0xf);
    x += DPP0(x, 0x142, 0xa); x += DPP0(x, 0x143, 0xc);
    return x;
}
__device__ __forceinline__ u32 umax(u32 a, u32 b) { return a > b ? a : b; }
__device__ __forceinline__ u32 wave_max(u32 x) {
    x = umax(x, DPP0(x, 0x111, 0xf)); x = umax(x, DPP0(x, 0x112, 0xf)); x = umax(x, DPP0(x, 0x114, 0xf)); x = umax(x, DPP0(x, 0x118, 0xf));
    x = umax(x, DPP0(x, 0x142, 0xa)); x = umax(x, DPP0(x, 0x143, 0xc));
    return (u32)__builtin_amdgcn_readlane((int)x, 63);
}
#define WAIT_VM(n) asm volatile("s_waitcnt vmcnt(" #n ")" ::: "memory")
#define WAIT_LGKM0() asm volatile("s_waitcnt lgkmcnt(0)" ::: "memory")
#define CBAR() asm volatile("" ::: "memory")
#define BAR() asm volatile("s_waitcnt lgkmcnt(0)\n\ts_barrier" ::: "memory")

struct Ctx {
    char* ws;
    char* lws;
    u32 call;
    int rank;
};

__device__ __forceinline__ bool tmo_hit(const Ctx& c) {
    return rfl(__hip_atomic_load((GAS u32*)(c.ws + OFF_TMO), __ATOMIC_RELAXED, __HIP_MEMORY_SCOPE_AGENT)) == c.call;
}
__device__ __forceinline__ bool spin_fail(const Ctx& c, u64 t0) {
    if (tmo_hit(c) || __builtin_amdgcn_s_memrealtime() - t0 > DEADLINE) {
        __hip_atomic_store((GAS u32*)(c.ws + OFF_TMO), c.call, __ATOMIC_RELAXED, __HIP_MEMORY_SCOPE_AGENT);
        return true;
    }
    __builtin_amdgcn_s_sleep(2);
    return false;
}
__device__ __forceinline__ u64 load_rec(const char* p) {
    return __hip_atomic_load((GAS u64*)p, __ATOMIC_RELAXED, __HIP_MEMORY_SCOPE_SYSTEM);
}
__device__ __forceinline__ u32 wait_rec(const Ctx& c, const char* p, u64 first) {
    u32 hi = rfl((u32)(first >> 32)), lo = rfl((u32)first);
    if (hi == c.call) return lo;
    const u64 t0 = __builtin_amdgcn_s_memrealtime();
    for (u32 n = 1;; ++n) {
        const u64 r = load_rec(p);
        hi = rfl((u32)(r >> 32)); lo = rfl((u32)r);
        if (hi == c.call) return lo;
        if ((n & 7u) == 0u && spin_fail(c, t0)) return 0u;
    }
}
__device__ __forceinline__ bool wait_cnt(const Ctx& c, const char* p, u32 target) {
    const u64 t0 = __builtin_amdgcn_s_memrealtime();
    for (u32 n = 1;; ++n) {
        if (__all(__hip_atomic_load((GAS u32*)p, __ATOMIC_RELAXED, __HIP_MEMORY_SCOPE_AGENT) >= target)) return true;
        if ((n & 7u) == 0u && spin_fail(c, t0)) return false;
        __builtin_amdgcn_s_sleep(100);
    }
}

constexpr u32 IMG_LEN = 4096, IMG_STR = 4160, INFO_END = 0xFFFFFFFFu;

__device__ __forceinline__ u32 pack_unit(const u32x4 (&v)[8], char* slot, LAS char* lds, LAS char* img, int wid, int mate, bool back, bool is_input, bool force_raw, int lane) {
    LAS u32* xme = (LAS u32*)(lds + L_XCH) + wid * 64 + lane;
    LAS u32* xot = (LAS u32*)(lds + L_XCH) + mate * 64 + lane;
    u32 info = USLOT / 128;
    if (force_raw) {
        const rsrc_t dst = mk_rsrc(slot, USLOT);
#pragma unroll
        for (int g = 0; g < 8; ++g) __builtin_amdgcn_raw_buffer_store_b128(v[g], dst, (back ? 8192u : 0u) + (u32)lane * 16u + g * 1024u, 0, AUX_SYS);
        WAIT_VM(0);
        BAR();
        return info;
    }
#pragma unroll
    for (int i = 0; i < 8; ++i) *(LAS u32x4*)(img + (back ? 8192 : 0) + i * 1024 + lane * 16) = (u32x4){0u, 0u, 0u, 0u};
    u32 delta = 0;
    if (is_input) {
        u32 a = 0;
#pragma unroll
        for (int g = 0; g < 8; ++g)
#pragma unroll
            for (int j = 0; j < 4; ++j) a += (v[g][j] >> 7) & 0x00FF00FFu;
        a = (a & 0xFFFFu) + (a >> 16);
        *xme = a;
        BAR();
        const u32 tot = (u32)__builtin_amdgcn_readlane((int)wave_scan_add(a + *xot), 63);
        delta = (u32)(((int)tot - 1024656) >> 13) & 0xFFu;
        BAR();
    }
    const u32 D2 = ((delta << 7) & 0xFFFFu) * 0x00010001u;
    const u32 tab = (u32)(size_t)lds + L_ENC;
    u32 e[32], tot = 0, mn = 15;
#pragma unroll
    for (int i = 0; i < 32; ++i) {
        const u32 t = pk_sub16(v[i >> 2][i & 3], D2);
        const u32 e0 = *(LAS u16*)(size_t)(((t >> 3) & 0x1FFEu) | tab), e1 = *(LAS u16*)(size_t)(((t >> 19) & 0x1FFEu) | tab);
        e[i] = e0 | (e1 << 16);
        const u32 l0 = e0 & 15u, l1 = e1 & 15u;
        tot += l0 + l1;
        mn = mn < l0 ? mn : l0; mn = mn < l1 ? mn : l1;
    }
    *xme = tot | (mn == 0u ? 0x10000u : 0u);
    BAR();
    const u32 xo = *xot;
    const bool rl = mn == 0u || (xo >> 16) != 0u;
    if (__any(rl)) {
#pragma unroll
        for (int i = 0; i < 32; ++i) { const u32 t = (pk_sub16(v[i >> 2][i & 3], D2) & 0xFFF0FFF0u) | 0x000C000Cu; e[i] = rl ? t : e[i]; }
    }
    const u32 len = rl ? 1536u : tot + (xo & 0xFFFFu);
    const u32 incl = wave_scan_add(len);
    const u32 bits = (u32)__builtin_amdgcn_readlane((int)incl, 63);
    const u32 L0 = 2047u - wave_max(rl ? 0u : 2047u - len);
    const u32 hi = wave_max(rl ? 0u : len);
    const u32 size = (IMG_STR + ((bits + 7u) >> 3) + 127u) & ~127u;
    const bool raw = size >= USLOT || (hi > L0 + 254u);
    if (!raw) {
#pragma unroll
        for (int h = 0; h < 2; ++h) {
            u32x4 n4;
#pragma unroll
            for (int g = 0; g < 4; ++g) {
                const u32x4 d = v[4 * h + g];
                n4[g] = (d[0] & 0x000F000Fu) | ((d[1] & 0x000F000Fu) << 4) | ((d[2] & 0x000F000Fu) << 8) | ((d[3] & 0x000F000Fu) << 12);
            }
            *(LAS u32x4*)(img + (back ? 2048 : 0) + h * 1024 + lane * 16) = n4;
        }
        if (!back) *(LAS unsigned char*)(img + IMG_LEN + lane) = (unsigned char)(rl ? 255u : len - L0);
        const u32 start = back ? incl : incl - len;
        u32 cnt = back ? (0u - start) & 31u : start & 31u, acc = 0;
        u32 ptr = (u32)(size_t)(img + IMG_STR) + (back ? ((start + cnt) >> 5) - 1u : start >> 5) * 4u;
        const u32 step = back ? (u32)-4 : 4u, ptr0 = ptr, dummy = (u32)(size_t)lds + L_DUMMY + (u32)lane * 4u;
        u32 w0 = 0;
#pragma unroll
        for (int i = 0; i < 32; ++i) {
            const u32 l0 = e[i] & 15u, l1 = (e[i] >> 16) & 15u;
            const u32 c01 = ((e[i] >> 4) & 0xFFFu) | ((e[i] >> 20) << l0);
            acc |= c01 << cnt;
            const u32 over = c01 >> ((32u - cnt) & 31u);
            cnt += l0 + l1;
            const bool full = cnt >= 32u;
            const bool first = ptr == ptr0;
            *(LAS u32*)(size_t)((full && !first) ? ptr : dummy) = back ? __builtin_bitreverse32(acc) : acc;
            w0 = (full && first) ? acc : w0;
            acc = full ? over : acc;
            ptr += full ? step : 0u;
            cnt &= 31u;
        }
        __hip_atomic_fetch_or((LAS u32*)(size_t)ptr0, back ? __builtin_bitreverse32(w0) : w0, __ATOMIC_RELAXED, __HIP_MEMORY_SCOPE_WORKGROUP);
        __hip_atomic_fetch_or((LAS u32*)(size_t)ptr, back ? __builtin_bitreverse32(acc) : acc, __ATOMIC_RELAXED, __HIP_MEMORY_SCOPE_WORKGROUP);
        info = (size >> 7) | ((is_input ? 1u : 0u) << 8) | (L0 << 9) | (delta << 20);
    } else {
        const rsrc_t dst = mk_rsrc(slot, USLOT);
#pragma unroll
        for (int g = 0; g < 8; ++g) __builtin_amdgcn_raw_buffer_store_b128(v[g], dst, (back ? 8192u : 0u) + (u32)lane * 16u + g * 1024u, 0, AUX_SYS);
    }
    BAR();
    if (!raw) {
        const rsrc_t dst = mk_rsrc(slot, size);
#pragma unroll
        for (int i = 0; i < 8; ++i) {
            const u32 o = (2u * i + (back ? 1u : 0u)) * 1024u;
            if (o < size) __builtin_amdgcn_raw_buffer_store_b128(*(LAS u32x4*)(img + o + lane * 16), dst, (u32)lane * 16u + o, 0, AUX_SYS);
        }
    }
    WAIT_VM(0);
    BAR();
    return info;
}

__device__ __forceinline__ char* peer_ws(const Params& p, u32 q) { return (char*)p.ws_all[q < NR ? q : 0]; }
__device__ __forceinline__ size_t f1_off(u32 call, u32 src, u32 s) { return OFF_F1 + (size_t)(((call & 1u) * NR + src) * NU + s) * F1_STRIDE; }

__device__ __forceinline__ void publish(const Ctx& c, u32 s, u32 info, u32 dest, int lane) {
    WAIT_VM(0);
    const bool act = dest < NR ? lane == (int)dest : (lane < NR && lane != c.rank);
    if (act) __hip_atomic_store((GAS u64*)(c.lws + f1_off(c.call, c.rank, s)), ((u64)c.call << 32) | info, __ATOMIC_RELAXED, __HIP_MEMORY_SCOPE_SYSTEM);
}
__device__ __forceinline__ void count_panel(const Ctx& c, int idx, int lane) {
    if (lane == 0) __hip_atomic_fetch_add((GAS u32*)(c.ws + OFF_F3 + (size_t)idx * 128), 1u, __ATOMIC_RELAXED, __HIP_MEMORY_SCOPE_AGENT);
}

struct UnitDst { u32 off, gstride; int kind; int cnt; };
__device__ __forceinline__ size_t unit_src(u32 call, u32 s, u32 me) {
    size_t o = OFF_ENC + (size_t)(call & 1u) * ENC_PAR;
    if (s < NIU) {
        const u32 code = ORD.slot[s >> 3], r = s & 7u;
        o += code < 32u ? ENC_X + (size_t)(8u * code + r) * USLOT : ENC_W + (size_t)(me * NWU + 8u * (code - 32u) + r) * USLOT;
    } else o += ENC_O + (size_t)(s - NIU) * USLOT;
    return o;
}
__device__ __forceinline__ UnitDst unit_dst(u32 s, u32 half, u32 src, int lane) {
    UnitDst d;
    if (s < NIU) {
        const u32 code = ORD.slot[s >> 3], r = s & 7u;
        if (code < 32u) {
            d.kind = 0; d.cnt = (int)(code >> 1); d.gstride = KT * 2;
            d.off = (2u * (8u * code + r) + half) * 8u * (KT * 2) + src * (KS * 2) + (u32)lane * 16u;
        } else {
            const u32 j = code - 32u;
            d.kind = 1; d.cnt = 16 + (int)j; d.gstride = 64;
            d.off = (128u * j + 16u * r + ((u32)lane & 15u)) * (KT * 2) + (src * KS + half * 256u + ((u32)lane >> 4) * 8u) * 2u;
        }
    } else {
        const u32 o = s - NIU;
        u32 i, j; tile_of(o >> 2, i, j);
        d.kind = 2; d.cnt = -1; d.gstride = 4u * NTOT * 2u;
        d.off = (256u * i + 64u * (o & 3u) + 32u * half + ((u32)lane >> 4)) * (NTOT * 2u) + (src * NS + 128u * j + ((u32)lane & 15u) * 8u) * 2u;
    }
    return d;
}

__device__ __forceinline__ u32 take_ticket(rsrc_t ctr, int lane) {
    return (u32)__builtin_amdgcn_raw_ptr_buffer_atomic_add_i32(1, ctr, lane == 0 ? 0 : (int)0x80000000u, 0, 0);
}
__device__ __forceinline__ u32 lds_ld(LAS char* lds, int off) { return *(volatile LAS u32*)(lds + off); }
__device__ __forceinline__ void lds_st(LAS char* lds, int off, u32 v) { *(volatile LAS u32*)(lds + off) = v; }

#define STR2(x) #x
#define STR(x) STR2(x)
__device__ __forceinline__ void dma_kib(u32x4 rs, u32 voff, u32 lds_addr) {
    u32 keep;
    asm volatile("s_nop 4\n\ts_mov_b32 %0, m0\n\ts_mov_b32 m0, %3\n\ts_nop 0\n\tbuffer_load_dwordx4 %1, %2, 0 offen sc0 sc1 lds\n\ts_mov_b32 m0, %0"
                 : "=&s"(keep) : "v"(voff), "s"(rs), "s"(lds_addr) : "memory");
}

__device__ __forceinline__ void sched_role(const Ctx& c, int cw, LAS char* lds, int lane) {
    const int link = cw % 7;
    const u32 peer = (u32)(c.rank + 1 + link) & 7u;
    const char* f1 = c.ws + f1_off(c.call, peer, 0);
    const rsrc_t rT = mk_rsrc(c.ws + OFF_TK + (size_t)((c.call & 1u) * 8u + (u32)link) * 128, 4);
    constexpr u32 NFIX = 2;
    u32 s = (u32)(cw / 7);
    for (u32 j = 0;; ++j) {
        u32 info = INFO_END;
        if (s < NU) info = wait_rec(c, f1 + (size_t)s * F1_STRIDE, load_rec(f1 + (size_t)s * F1_STRIDE));
        const u32 q = j & 7u;
        lds_st(lds, L_QINFO + q * 4, info); lds_st(lds, L_QS + q * 4, s); lds_st(lds, L_QFULL + q * 4, j + 1);
        if (info == INFO_END) break;
        while (rfl(lds_ld(lds, L_QUSED)) + RUNAHEAD < j + 2) __builtin_amdgcn_s_sleep(2);
        if (j + 1 < NFIX) s += WPL;
        else s = rfl(take_ticket(rT, lane)) + NFIX * WPL;
    }
}

__device__ __forceinline__ void loader_role(const Params& p, const Ctx& c, int cw, LAS char* lds, int lane) {
    const u32 peer = (u32)(c.rank + 1 + cw % 7) & 7u;
    const char* pws = peer_ws(p, peer);
    bool pending = false;
    u32 pinfo = 0, ps = 0, k = 0;
    auto publish_prev = [&]() {
        const u32 sl = (k - 1u) & (NSLOT - 1);
        lds_st(lds, L_INFO + sl * 4, pinfo); lds_st(lds, L_SIDX + sl * 4, ps); lds_st(lds, L_FULL + sl * 4, k);
        pending = false;
    };
    for (;; ++k) {
        const u32 q = k & 7u, sl = k & (NSLOT - 1), need = 2u * (k / NSLOT);
        if (rfl(lds_ld(lds, L_QFULL + q * 4)) != k + 1 || rfl(lds_ld(lds, L_FREE + sl * 4)) < need) {
            if (pending) { WAIT_VM(0); publish_prev(); }
            while (rfl(lds_ld(lds, L_QFULL + q * 4)) != k + 1) __builtin_amdgcn_s_sleep(1);
            if (rfl(lds_ld(lds, L_QINFO + q * 4)) != INFO_END)
                while (rfl(lds_ld(lds, L_FREE + sl * 4)) < need) __builtin_amdgcn_s_sleep(1);
        }
        const u32 info = rfl(lds_ld(lds, L_QINFO + q * 4)), s = rfl(lds_ld(lds, L_QS + q * 4));
        lds_st(lds, L_QUSED, k + 1);
        if (info == INFO_END) break;
        const u32 size = (info & 0xFFu) * 128u;
        const u64 a = (u64)(pws + unit_src(c.call, s, c.rank));
        const u32x4 rs = {rfl((u32)a), rfl((u32)(a >> 32)) & 0xFFFFu, size, 0x00020000u};
        const u32 dst = (u32)(size_t)lds + L_SLOT + sl * 16384u;
#pragma unroll
        for (int j = 0; j < 16; ++j) {
            if (j * 1024u < size) {
                asm volatile("s_waitcnt vmcnt(" STR(WIN) " - 1)" ::: "memory");
                if (j == WIN - 1 && pending) publish_prev();
                dma_kib(rs, (u32)lane * 16u + j * 1024u, dst + j * 1024u);
            }
        }
        if (pending) { WAIT_VM(0); publish_prev(); }
        pending = true; pinfo = info; ps = s;
    }
    if (pending) { WAIT_VM(0); publish_prev(); }
    for (u32 j = 0; j < 4; ++j) {
        const u32 kk = k + j, sl = kk & (NSLOT - 1);
        if (kk >= NSLOT) while (rfl(lds_ld(lds, L_FREE + sl * 4)) < 2u * (kk / NSLOT)) __builtin_amdgcn_s_sleep(1);
        lds_st(lds, L_INFO + sl * 4, INFO_END); lds_st(lds, L_FULL + sl * 4, kk + 1);
    }
}

template <bool BACK>
__device__ __forceinline__ void unpack_half(LAS char* lds, LAS char* slot, u32 info, rsrc_t rD, const UnitDst& d, int lane) {
    const u32x4 n0 = *(LAS u32x4*)(slot + (BACK ? 2048 : 0) + lane * 16), n1 = *(LAS u32x4*)(slot + (BACK ? 3072 : 1024) + lane * 16);
    const u32 lb = *(LAS unsigned char*)(slot + IMG_LEN + lane);
    const u32 len = lb == 255u ? 1536u : ((info >> 9) & 0x7FFu) + lb;
    u32 pos = wave_scan_add(len) - (BACK ? 32u : len) + ((u32)(size_t)slot + IMG_STR) * 8u;
    const u32 lut = (u32)(size_t)lds + (lb == 255u ? (u32)L_DEC_ID : ((info >> 8) & 1u) ? (u32)L_DEC_IN : (u32)L_DEC_OUT);
    const u32 D2 = ((((info >> 20) & 0xFFu) << 7) & 0xFFFFu) * 0x00010001u;
#pragma unroll
    for (int g = 0; g < 8; ++g) {
        const u32 N = g < 4 ? n0[g & 3] : n1[g & 3];
        u32x4 v;
#pragma unroll
        for (int q = 0; q < 4; ++q) {
            LAS u32* wp = (LAS u32*)(size_t)((pos >> 3) & ~3u);
            u32 win = __builtin_amdgcn_alignbit(wp[1], wp[0], pos);
            if (BACK) win = __builtin_bitreverse32(win);
            const u32 e0 = *(LAS u16*)(size_t)(lut + ((win & 0xFFFu) << 1));
            const u32 l0 = e0 & 15u;
            win >>= l0;
            const u32 e1 = *(LAS u16*)(size_t)(lut + ((win & 0xFFFu) << 1));
            if (BACK) pos -= l0 + (e1 & 15u); else pos += l0 + (e1 & 15u);
            v[q] = bfi(0x000F000Fu, N >> (4 * q), pk_add16(perm(e1, e0, 0x05040100u), D2));
        }
        __builtin_amdgcn_raw_buffer_store_b128(v, rD, d.off + g * d.gstride, 0, AUX_DEV);
    }
}

__device__ __forceinline__ void unpack_role(const Params& p, const Ctx& c, int cw, LAS char* lds, int wid, int lane) {
    const u32 peer = (u32)(c.rank + 1 + cw % 7) & 7u;
    for (u32 h = (u32)wid - 2u;; h += 6u) {
        const u32 k = h >> 1, half = h & 1u, sl = k & (NSLOT - 1);
        while (rfl(lds_ld(lds, L_FULL + sl * 4)) != k + 1) __builtin_amdgcn_s_sleep(1);
        const u32 info = rfl(lds_ld(lds, L_INFO + sl * 4));
        if (info == INFO_END) break;
        const u32 s = rfl(lds_ld(lds, L_SIDX + sl * 4));
        CBAR();
        const UnitDst d = unit_dst(s, half, peer, lane);
        const rsrc_t rD = mk_rsrc(d.kind == 2 ? (char*)p.out : c.ws + (d.kind == 0 ? OFF_A : OFF_B), d.kind == 2 ? 256u << 20 : 32u << 20);
        LAS char* slot = lds + L_SLOT + sl * 16384;
        if (info == 0u || (info & 0xFFu) == USLOT / 128) {
#pragma unroll
            for (int g = 0; g < 8; ++g) {
                u32x4 v = *(LAS u32x4*)(slot + half * 8192 + g * 1024 + lane * 16);
                if (info == 0u) v = (u32x4){0x7FC07FC0u, 0x7FC07FC0u, 0x7FC07FC0u, 0x7FC07FC0u};
                __builtin_amdgcn_raw_buffer_store_b128(v, rD, d.off + g * d.gstride, 0, AUX_DEV);
            }
        } else if (half) unpack_half<true>(lds, slot, info, rD, d, lane);
        else unpack_half<false>(lds, slot, info, rD, d, lane);
        WAIT_LGKM0();
        if (lane == 0) __hip_atomic_fetch_add((LAS u32*)(lds + L_FREE + sl * 4), 1u, __ATOMIC_RELAXED, __HIP_MEMORY_SCOPE_WORKGROUP);
        if (d.cnt >= 0 && info != 0u) { WAIT_VM(0); count_panel(c, d.cnt, lane); }
    }
}

__device__ __forceinline__ u32 job_entry(int J) {
    return J < 16 ? ((u32)(J >> 1) << 16 | (u32)(J >> 1) << 8 | 8u << 1 | (u32)(J & 1)) : ORD.job[J];
}
static_assert(ORD_H.job[15] == (7 << 8 | 8 << 1 | 1) && ORD_H.slot[7] == 7 && ORD_H.job[6] == (3 << 8 | 8 << 1 | 0) && ORD_H.slot[3] == 3, "order table");
__device__ __forceinline__ void pack_load(const Params& p, int J, u32x4 (&v)[8], int wid, int lane) {
    const u32 e = job_entry(J), code = e >> 16, dest = (e >> 1) & 15u, sub = e & 1u;
    if (dest == 8u) {
        const u32 xu = 16u * code + 8u * sub + (u32)wid;
        const rsrc_t rx = mk_rsrc(p.x, M * KS * 2);
#pragma unroll
        for (int g = 0; g < 8; ++g) v[g] = __builtin_amdgcn_raw_buffer_load_b128(rx, (xu * 8u + g) * (KS * 2) + (u32)lane * 16u, 0, 0);
    } else {
        const u32 col0 = dest * NS + 128u * (code - 32u) + 64u * sub;
        const rsrc_t rw = mk_rsrc(p.w, KS * NTOT * 2);
#pragma unroll
        for (int i = 0; i < 8; ++i) v[i] = __builtin_amdgcn_raw_buffer_load_b128(rw, ((u32)wid * 64u + i * 8u + ((u32)lane >> 3)) * (NTOT * 2) + col0 * 2u + ((u32)lane & 7u) * 16u, 0, 0);
    }
}
__device__ __forceinline__ void pack_job(const Ctx& c, int J, u32x4 (&v)[8], LAS char* lds, int wid, int lane) {
    const u32 e = job_entry(J), q = (e >> 8) & 255u, code = e >> 16, dest = (e >> 1) & 15u, sub = e & 1u;
    char* enc = c.ws + OFF_ENC + (size_t)(c.call & 1u) * ENC_PAR;
    if (dest == 8u) {
        const u32 xh = 16u * code + 8u * sub + (u32)wid, r = 4u * sub + ((u32)wid >> 1);
        const rsrc_t rA = mk_rsrc(c.ws + OFF_A, 32u << 20);
        const u32 info = pack_unit(v, enc + ENC_X + (size_t)(8u * code + r) * USLOT, lds, lds + L_SCR_W + (wid >> 1) * 16384, wid, wid ^ 1, (wid & 1) != 0, true, 8u * q + r < HEAD_RAW, lane);
        if (!(wid & 1)) publish(c, 8u * q + r, info, NR, lane);
#pragma unroll
        for (int g = 0; g < 8; ++g) __builtin_amdgcn_raw_buffer_store_b128(v[g], rA, (xh * 8u + g) * (KT * 2) + (u32)c.rank * (KS * 2) + (u32)lane * 16u, 0, AUX_DEV);
        WAIT_VM(0);
        count_panel(c, (int)(code >> 1), lane);
    } else {
        BAR();
#pragma unroll
        for (int i = 0; i < 8; ++i) *(LAS u32x4*)(lds + (wid * 64 + i * 8 + (lane >> 3)) * WT_STRIDE + (lane & 7) * 16) = v[i];
        BAR();
        const int nq = wid & 3, kh = wid >> 2, kq = lane >> 4, qq = (lane >> 2) & 3, pp = lane & 3;
#pragma unroll
        for (int g = 0; g < 8; ++g) {
            const int k0 = kh * 256 + 32 * g + 8 * kq;
            const s16x4 a = __builtin_amdgcn_ds_read_tr16_b64_v4i16((LAS s16x4*)(lds + (k0 + qq) * WT_STRIDE + (16 * nq + 4 * pp) * 2));
            const s16x4 b = __builtin_amdgcn_ds_read_tr16_b64_v4i16((LAS s16x4*)(lds + (k0 + 4 + qq) * WT_STRIDE + (16 * nq + 4 * pp) * 2));
            const u32x2 a2 = __builtin_bit_cast(u32x2, a), b2 = __builtin_bit_cast(u32x2, b);
            v[g] = (u32x4){a2.x, a2.y, b2.x, b2.y};
        }
        const u32 j = code - 32u, r = sub * 4u + (u32)nq;
        if (dest == (u32)c.rank) {
            const rsrc_t rB = mk_rsrc(c.ws + OFF_B, 32u << 20);
            const u32 off = (128u * j + 16u * r + ((u32)lane & 15u)) * (KT * 2) + ((u32)c.rank * KS + (u32)kh * 256u + (u32)kq * 8u) * 2u;
#pragma unroll
            for (int g = 0; g < 8; ++g) __builtin_amdgcn_raw_buffer_store_b128(v[g], rB, off + g * 64u, 0, AUX_DEV);
            WAIT_VM(0);
            count_panel(c, 16 + (int)j, lane);
        } else {
            const u32 info = pack_unit(v, enc + ENC_W + (size_t)(dest * NWU + 8u * j + r) * USLOT, lds, lds + L_SCR_W + nq * 16384, wid, wid ^ 4, kh != 0, true, false, lane);
            if (!kh) publish(c, 8u * q + r, info, dest, lane);
        }
    }
}

__device__ __forceinline__ float gelu_f(float y) {
    const float u = y * (1.0f + 0.044715f * y * y) * (-2.0f * 0.7978845608028654f * 1.4426950408889634f);
    return y * __builtin_amdgcn_rcpf(1.0f + __builtin_amdgcn_exp2f(u));
}
__device__ __forceinline__ u32 cvt_pk_bf16(float lo, float hi) { u32 r; asm("v_cvt_pk_bf16_f32 %0, %1, %2" : "=v"(r) : "v"(lo), "v"(hi)); return r; }

__device__ __forceinline__ void gemm_tile(const Params& p, const Ctx& c, int k_order, LAS char* lds, int wid, int lane) {
    u32 uti, utj; tile_of((u32)k_order, uti, utj);
    const int ti = (int)uti, tj = (int)utj;
    LAS u32* okw = (LAS u32*)(lds + L_OK);
    if (wid == 0) {
        const int l7 = lane & 7;
        const int idx = l7 < 4 ? (ti & ~3) + l7 : 16 + (tj & ~3) + (l7 - 4);
        const bool ok = wait_cnt(c, c.ws + OFF_F3 + (size_t)idx * 128, c.call * (l7 < 4 ? 256u : 128u));
        if (lane == 0) *okw = ok ? 1u : 0u;
    }
    __syncthreads();
    const bool ok = rfl(*okw) != 0u;

    const char* gA = c.ws + OFF_A + (size_t)ti * BM * (KT * 2);
    const char* gB = c.ws + OFF_B + (size_t)tj * BN * (KT * 2);
    const u32 lsrc = (u32)(lane >> 3) * (KT * 2) + (u32)((lane & 7) ^ (lane >> 3)) * 16u;
    const char* sA = gA + (size_t)(wid * 32) * (KT * 2) + lsrc;
    const char* sB = gB + (size_t)(wid * 16) * (KT * 2) + lsrc;
    auto stage = [&](int kt, int buf) {
        LAS char* base = lds + buf * LDS_STAGE;
#pragma unroll
        for (int i = 0; i < 4; ++i)
            __builtin_amdgcn_global_load_lds((const GAS u32*)(sA + (size_t)i * 8 * (KT * 2) + kt * (BK * 2)), (LAS u32*)(base + (wid * 4 + i) * 1024), 16, 0, AUX_DEV);
#pragma unroll
        for (int i = 0; i < 2; ++i)
            __builtin_amdgcn_global_load_lds((const GAS u32*)(sB + (size_t)i * 8 * (KT * 2) + kt * (BK * 2)), (LAS u32*)(base + BM * BK * 2 + (wid * 2 + i) * 1024), 16, 0, AUX_DEV);
    };
    const int wm = wid >> 1, wn = wid & 1, fr = lane & 15, fq = lane >> 4;
    f32x4 acc[4][4];
#pragma unroll
    for (int m = 0; m < 4; ++m)
#pragma unroll
        for (int n = 0; n < 4; ++n) acc[m][n] = (f32x4){0.f, 0.f, 0.f, 0.f};
    const int aoff0 = (wm * 64 + fr) * 128 + ((fq ^ (fr & 7)) * 16), aoff1 = (wm * 64 + fr) * 128 + (((4 + fq) ^ (fr & 7)) * 16);
    const int boff0 = BM * BK * 2 + (wn * 64 + fr) * 128 + ((fq ^ (fr & 7)) * 16), boff1 = BM * BK * 2 + (wn * 64 + fr) * 128 + (((4 + fq) ^ (fr & 7)) * 16);
    stage(0, 0); stage(1, 1);
    int buf = 0;
    for (int kt = 0; kt < NKT; ++kt) {
        if (kt + 1 < NKT) { WAIT_VM(6); } else { WAIT_VM(0); }
        WAIT_LGKM0();
        __builtin_amdgcn_s_barrier();
        CBAR();
        LAS char* base = lds + buf * LDS_STAGE;
        bf16x8 a0[4], b0[4], a1[4], b1[4];
#pragma unroll
        for (int n = 0; n < 4; ++n) b0[n] = *(LAS bf16x8*)(base + boff0 + n * 2048);
#pragma unroll
        for (int m = 0; m < 4; ++m) a0[m] = *(LAS bf16x8*)(base + aoff0 + m * 2048);
        __builtin_amdgcn_sched_barrier(0);
#pragma unroll
        for (int n = 0; n < 4; ++n) b1[n] = *(LAS bf16x8*)(base + boff1 + n * 2048);
#pragma unroll
        for (int m = 0; m < 4; ++m) a1[m] = *(LAS bf16x8*)(base + aoff1 + m * 2048);
        __builtin_amdgcn_sched_barrier(0);
        __builtin_amdgcn_s_setprio(1);
#pragma unroll
        for (int m = 0; m < 4; ++m)
#pragma unroll
            for (int n = 0; n < 4; ++n) acc[m][n] = __builtin_amdgcn_mfma_f32_16x16x32_bf16(b0[n], a0[m], acc[m][n], 0, 0, 0);
        __builtin_amdgcn_sched_barrier(0);
        WAIT_LGKM0();
        if (kt + 2 < NKT) stage(kt + 2, buf == 0 ? 2 : buf - 1);
        __builtin_amdgcn_sched_barrier(0);
#pragma unroll
        for (int m = 0; m < 4; ++m)
#pragma unroll
            for (int n = 0; n < 4; ++n) acc[m][n] = __builtin_amdgcn_mfma_f32_16x16x32_bf16(b1[n], a1[m], acc[m][n], 0, 0, 0);
        __builtin_amdgcn_s_setprio(0);
        __builtin_amdgcn_sched_barrier(0);
        buf = buf == 2 ? 0 : buf + 1;
    }
    __syncthreads();
#pragma unroll
    for (int m = 0; m < 4; ++m)
#pragma unroll
        for (int n = 0; n < 4; ++n) {
            const f32x4 a = acc[m][n];
            u32x2 w2 = {cvt_pk_bf16(gelu_f(a[0]), gelu_f(a[1])), cvt_pk_bf16(gelu_f(a[2]), gelu_f(a[3]))};
            if (!ok) w2 = (u32x2){0x7FC07FC0u, 0x7FC07FC0u};
            *(LAS u32x2*)(lds + (wm * 64 + m * 16 + fr) * EP_STRIDE + (wn * 64 + n * 16 + fq * 4) * 2) = w2;
        }
    __syncthreads();
    u32x4 v[8];
#pragma unroll
    for (int g = 0; g < 8; ++g) v[g] = *(LAS u32x4*)(lds + (wid * 32 + 4 * g + (lane >> 4)) * EP_STRIDE + (lane & 15) * 16);
    if (ok) {
        const u32 o = (u32)k_order * 4u + ((u32)wid >> 1);
        const u32 info = pack_unit(v, c.ws + OFF_ENC + (size_t)(c.call & 1u) * ENC_PAR + ENC_O + (size_t)o * USLOT, lds, lds + L_SCR_EP + (wid >> 1) * 16384,
                                   wid, wid ^ 1, (wid & 1) != 0, false, o >= NOU - TAIL_RAW, lane);
        if (!(wid & 1)) publish(c, NIU + o, info, NR, lane);
    }
    const rsrc_t rO = mk_rsrc(p.out, 256u << 20);
    const u32 ooff = (u32)(ti * BM + wid * 32 + (lane >> 4)) * (NTOT * 2u) + (u32)(c.rank * NS + tj * BN + (lane & 15) * 8) * 2u;
#pragma unroll
    for (int g = 0; g < 8; ++g) __builtin_amdgcn_raw_buffer_store_b128(v[g], rO, ooff + g * (4u * NTOT * 2u), 0, AUX_DEV);
    __syncthreads();
}

__device__ __forceinline__ void load_table(LAS char* dst, const u16* src, int tid) {
    *(LAS u32x4*)(dst + tid * 16) = *(const u32x4*)((const char*)src + tid * 16);
}

__global__ void __launch_bounds__(NTHR, 2) fused_kernel(Params p) {
    extern __shared__ __attribute__((aligned(16))) char smem[];
    LAS char* lds = (LAS char*)smem;
    const int tid = threadIdx.x, lane = tid & 63, wid = (int)rfl(tid >> 6), b = blockIdx.x;
    constexpr int NPULL = 7 * WPL, NPACK = NWG - NPULL;
    const bool puller = b < NPULL;
    const int pj = b - NPULL;
    u32x4 v[8];
    if (!puller) pack_load(p, pj, v, wid, lane);
    Ctx c;
    c.rank = p.rank; c.ws = peer_ws(p, (u32)p.rank); c.lws = peer_ws(p, (u32)lane & 7u);
    LAS u32* cw = (LAS u32*)(lds + L_CALL);
    if (tid == 0) *cw = __hip_atomic_load((GAS u32*)(c.ws + OFF_CALL), __ATOMIC_RELAXED, __HIP_MEMORY_SCOPE_AGENT) / NWG + 1u;
    if (puller) {
        load_table(lds + L_DEC_OUT, TAB.dec_out, tid);
        load_table(lds + L_DEC_IN, TAB.dec_in, tid);
#pragma unroll
        for (int i = 0; i < 8; ++i) *(LAS u16*)(lds + L_DEC_ID + (tid * 8 + i) * 2) = (u16)(((tid * 8 + i) << 4) | 12);
        if (tid < 96) *(LAS u32*)(lds + L_FULL + tid * 4) = 0u;
    } else load_table(lds + L_ENC, TAB.enc_in, tid);
    __syncthreads();
    c.call = rfl(*cw);
    if (b == 0 && tid < 8) __hip_atomic_store((GAS u32*)(c.ws + OFF_TK + (size_t)(((c.call + 1u) & 1u) * 8u + (u32)tid) * 128), 0u, __ATOMIC_RELAXED, __HIP_MEMORY_SCOPE_AGENT);
    if (puller) {
        if (wid == 0) loader_role(p, c, b, lds, lane);
        else if (wid == 1) sched_role(c, b, lds, lane);
        else unpack_role(p, c, b, lds, wid, lane);
    } else {
        constexpr int NEARLY = 32, NLATE = NPACK - NEARLY;
        const bool early = b >= NCOPYWG && (b & 7) < 2;
        const int li = b < NCOPYWG ? pj : (NCOPYWG - NPULL) + ((b - NCOPYWG) >> 3) * 6 + (b & 7) - 2;
        auto job_at = [&](int r) { return r < 2 ? pj + r * NPACK : (early ? NJOBS : 2 * NPACK + li + (r - 2) * NLATE); };
        for (int r = 0;; r += 2) {
            u32x4 v2[8];
            const int J0 = job_at(r), J1 = job_at(r + 1), J2 = job_at(r + 2);
            if (J1 < NJOBS) pack_load(p, J1, v2, wid, lane);
            pack_job(c, J0, v, lds, wid, lane);
            if (J1 >= NJOBS) break;
            if (J2 < NJOBS) pack_load(p, J2, v, wid, lane);
            pack_job(c, J1, v2, lds, wid, lane);
            if (J2 >= NJOBS) break;
        }
        if (b >= NCOPYWG) {
            const int gi = b - NCOPYWG;
            __syncthreads();
            load_table(lds + L_ENC, TAB.enc_out, tid);
            for (int r = 0; r < 4; ++r) gemm_tile(p, c, (r * 8 + (b & 7)) * 16 + (gi >> 3), lds, wid, lane);
        }
    }
    __syncthreads();
    if (tid == 0) __hip_atomic_fetch_add((GAS u32*)(c.ws + OFF_CALL), 1u, __ATOMIC_RELAXED, __HIP_MEMORY_SCOPE_AGENT);
}

}

extern "C" void kernel_launch(int rank, int nranks, ncclComm_t comm, hipStream_t stream,
                              void* const* d_in, const int* in_sizes, int n_in,
                              void* d_out, int out_size, const int* in_dtypes, int out_dtype,
                              void* const* d_out_all, void* const* d_ws_all, size_t ws_bytes) {
    (void)comm; (void)in_dtypes; (void)out_dtype; (void)d_out_all;
    if (nranks != NR || n_in != 2 || in_sizes[0] != M * KS || in_sizes[1] != KS * NTOT || out_size != M * NTOT || ws_bytes < WS_NEED) {
        fprintf(stderr, "kernel_launch: unexpected problem shape\n");
        return;
    }
    static bool init = false;
    if (!init) {
        HIP_CHECK(hipFuncSetAttribute((const void*)fused_kernel, hipFuncAttributeMaxDynamicSharedMemorySize, LDS_BYTES));
        init = true;
    }
    Params p;
    p.x = (const unsigned short*)d_in[0]; p.w = (const unsigned short*)d_in[1]; p.out = (unsigned short*)d_out;
    p.ws_all = d_ws_all; p.rank = rank; p.pad = 0;
    hipLaunchKernelGGL(fused_kernel, dim3(NWG), dim3(NTHR), LDS_BYTES, stream, p);
    HIP_CHECK(hipGetLastError());
}
```
